```python
import numpy as np

kernel_name = "gpu_gemm_ar_m8192_k8192_n8192_bf16_silu"


ABSMAX_RTOL = 0.008

COMPUTE_DTYPE = "bfloat16"
EPILOGUE = "silu"
IN_QUANT = None
OUT_QUANT = None
IN_DTYPES = {"x": "bfloat16", "w_mat": "bfloat16"}
WORLD_SIZE = 8

def _to_bf16(x):
    u = np.ascontiguousarray(x, dtype=np.float32).view(np.uint32)
    return ((u + 0x7FFF + ((u >> 16) & 1)) & 0xFFFF0000).view(np.float32)

def setup_inputs():

    x = _to_bf16(np.random.randn(8192, 8192).astype(np.float32))
    w_mat = _to_bf16((np.random.randn(8192, 8192) / np.sqrt(8192)).astype(np.float32))
    return {"x": x, "w_mat": w_mat}

def _epilogue(y):
    return (y / (1.0 + np.exp(-np.clip(y, -60.0, 60.0)))).astype(np.float32)

def reference(x, w_mat):
    y = (x @ w_mat).astype(np.float32)
    return _epilogue(y)
```

```cpp
#include <hip/hip_runtime.h>
#include <rccl/rccl.h>
#include <cstdio>

#define HIP_CHECK(expr) do { hipError_t _e = (expr); if (_e != hipSuccess) { \
    fprintf(stderr, "HIP error %s at %s:%d\n", hipGetErrorString(_e), __FILE__, __LINE__); \
    return; } } while (0)

namespace {

#define LAS __attribute__((address_space(3)))
typedef short bf16x8 __attribute__((ext_vector_type(8)));
typedef float f32x4 __attribute__((ext_vector_type(4)));
typedef unsigned u32x4 __attribute__((ext_vector_type(4)));
typedef __amdgpu_buffer_rsrc_t rsrc_t;

constexpr int NR = 8;
constexpr unsigned MiB = 1u << 20;
constexpr unsigned X_BYTES = 8192u * 1024u * 2u, W_BYTES = 1024u * 8192u * 2u, OUT_BYTES = 8192u * 8192u * 2u;
constexpr unsigned OFF_XS = 0, OFF_WS = 32 * MiB, OFF_PB = 64 * MiB, OFF_PL = 80 * MiB, OFF_FL = 96 * MiB;
constexpr unsigned WS_USED = 97 * MiB;
constexpr unsigned BLK_BYTES = 2 * MiB, KT_BYTES = 32768, HT_BYTES = 16384, UNIT_BYTES = 8192, TILE_BYTES = 131072;
constexpr unsigned F_CTR = 0, F_TMO = 128, F_LP = 1024, F_LF = 1536, F_FD = 2048, F_Q = 4096, F_FP = 8192, F_FR = 24576, F_BLK = 32768;
constexpr unsigned F_DBG = 262144;
constexpr int WPL = 8;
constexpr int NCOMM = 7 * WPL, NGEMM = 200, NWG = NCOMM + NGEMM;
#define NDIRECT 38
#define RDELAY 8
constexpr int NREL = 128 - NDIRECT;
constexpr unsigned TU = 16;
static_assert(((NREL + 5) / 6) * TU <= 256 && NDIRECT >= RDELAY, "relay flags");
constexpr unsigned long long DEADLINE = 100000000ull;
constexpr int LDS_BYTES = 131072 + 2048;
constexpr int AUX_WT = 17;

static constexpr unsigned char kColOrder[16] = {0,1,2,3,8,4,9,5,10,6,11,7,12,13,14,15};
constexpr unsigned long long pack_col_order() { unsigned long long v = 0; for (int i = 0; i < 16; ++i) v |= (unsigned long long)kColOrder[i] << (4 * i); return v; }
constexpr unsigned long long kColPacked = pack_col_order();
static constexpr unsigned char kTileRJ[128] = {0,8,1,9,16,17,2,10,24,25,18,26,32,33,34,3,40,41,42,11,48,49,50,19,56,57,58,27,64,65,66,35,72,73,74,43,51,59,67,75,80,81,82,83,88,89,90,91,4,12,20,28,36,44,52,60,68,76,84,92,96,97,98,99,100,5,13,21,29,37,45,53,61,69,77,85,93,101,104,105,106,107,108,109,6,14,22,30,38,46,54,62,70,78,86,94,102,110,112,113,114,115,116,117,118,7,15,23,31,39,47,55,63,71,79,87,95,103,111,119,120,121,122,123,124,125,126,127};
static constexpr unsigned short kTile1[200] = {0,21,128,52,134,139,95,182,1,22,129,53,135,142,175,183,2,36,132,144,138,146,96,184,3,24,136,54,84,150,176,185,4,25,140,141,85,154,177,186,5,26,48,145,86,158,178,187,6,37,49,148,75,87,179,192,7,28,50,149,143,88,180,199,10,29,51,55,147,89,181,200,8,30,130,69,151,90,190,201,9,38,131,70,169,91,191,202,11,32,133,71,59,92,97,203,15,33,137,72,76,93,193,204,19,34,65,152,172,162,194,205,23,39,66,153,173,94,195,210,27,40,67,56,64,155,196,211,12,41,68,73,77,159,197,217,13,42,62,156,188,163,198,218,14,43,63,157,189,164,208,219,31,44,78,57,82,165,209,220,16,45,79,74,83,166,212,221,17,46,80,160,206,167,213,222,18,47,81,161,207,170,214,223,35,60,98,58,226,171,215,224,20,61,99,168,227,174,216,225};
static constexpr unsigned short kTile2[200] = {105,239,65535,236,65535,65535,65535,65535,106,240,65535,237,65535,65535,65535,65535,107,241,65535,65535,65535,65535,65535,65535,108,243,65535,242,65535,65535,65535,65535,109,244,65535,65535,65535,65535,65535,65535,110,245,119,65535,65535,65535,65535,65535,111,246,233,65535,65535,65535,65535,65535,112,124,234,65535,65535,65535,65535,65535,113,252,235,247,65535,65535,65535,65535,114,125,65535,122,65535,65535,65535,65535,115,253,65535,250,65535,65535,65535,65535,116,127,65535,123,65535,65535,65535,65535,117,255,65535,251,65535,65535,65535,65535,118,65535,120,65535,65535,65535,65535,65535,100,65535,248,65535,65535,65535,65535,65535,228,65535,121,126,65535,65535,65535,65535,101,65535,249,254,65535,65535,65535,65535,229,65535,65535,65535,65535,65535,65535,65535,102,65535,65535,65535,65535,65535,65535,65535,230,65535,65535,65535,65535,65535,65535,65535,103,65535,65535,65535,65535,65535,65535,65535,231,65535,65535,65535,65535,65535,65535,65535,104,65535,65535,65535,65535,65535,65535,65535,232,65535,65535,65535,65535,65535,65535,65535,238,65535,65535,65535,65535,65535,65535,65535};

struct Params {
    const char* x; const char* w;
    char* out[NR]; char* ws[NR];
    int rank;
};

__device__ __forceinline__ rsrc_t mkrs(const void* p, unsigned n) { return __builtin_amdgcn_make_buffer_rsrc((void*)p, 0, n, 0x00020000); }
__device__ __forceinline__ u32x4 ld16(rsrc_t r, unsigned off) { return __builtin_amdgcn_raw_buffer_load_b128(r, off, 0, 0); }
__device__ __forceinline__ u32x4 ld16c(rsrc_t r, unsigned off) { return __builtin_amdgcn_raw_buffer_load_b128(r, off, 0, AUX_WT); }
__device__ __forceinline__ void st16(u32x4 v, rsrc_t r, unsigned off) { __builtin_amdgcn_raw_buffer_store_b128(v, r, off, 0, AUX_WT); }
#define WAIT_V(n) asm volatile("s_waitcnt vmcnt(" #n ")" ::: "memory")
#define WAIT_L(n) asm volatile("s_waitcnt lgkmcnt(" #n ")" ::: "memory")
#define BAR __builtin_amdgcn_s_barrier()
#define SCHED __builtin_amdgcn_sched_barrier(0)
__device__ __forceinline__ void wg_barrier() { asm volatile("s_waitcnt lgkmcnt(0)" ::: "memory"); __builtin_amdgcn_s_barrier(); asm volatile("" ::: "memory"); }
__device__ __forceinline__ unsigned rfl(unsigned v) { return (unsigned)__builtin_amdgcn_readfirstlane((int)v); }
__device__ __forceinline__ unsigned ld_flag(const unsigned* p) { return __hip_atomic_load(p, __ATOMIC_RELAXED, __HIP_MEMORY_SCOPE_SYSTEM); }
__device__ __forceinline__ void st_flag(unsigned* p, unsigned v) { __hip_atomic_store(p, v, __ATOMIC_RELAXED, __HIP_MEMORY_SCOPE_SYSTEM); }

#define STAMP(slot) do {} while (0)

__device__ __forceinline__ bool wave_wait(const unsigned* p, bool use, unsigned call, unsigned* tmo) {
    const unsigned long long t0 = __builtin_amdgcn_s_memrealtime();
    for (;;) {
        const unsigned v = use ? ld_flag(p) : call;
        if (__all(v >= call)) return true;
        if (rfl(__hip_atomic_load(tmo, __ATOMIC_RELAXED, __HIP_MEMORY_SCOPE_AGENT)) == call ||
            __builtin_amdgcn_s_memrealtime() - t0 > DEADLINE) {
            __hip_atomic_store(tmo, call, __ATOMIC_RELAXED, __HIP_MEMORY_SCOPE_AGENT);
            return false;
        }
        __builtin_amdgcn_s_sleep(4);
    }
}

template <int NW>
__device__ __forceinline__ bool wave_wait_n(const unsigned* p, unsigned call, unsigned* tmo) {
    const unsigned long long t0 = __builtin_amdgcn_s_memrealtime();
    for (;;) {
        unsigned v[NW];
#pragma unroll
        for (int k = 0; k < NW; ++k) v[k] = ld_flag(p + k * 32);
        bool all = true;
#pragma unroll
        for (int k = 0; k < NW; ++k) all = all && v[k] >= call;
        if (__all(all)) return true;
        if (rfl(__hip_atomic_load(tmo, __ATOMIC_RELAXED, __HIP_MEMORY_SCOPE_AGENT)) == call ||
            __builtin_amdgcn_s_memrealtime() - t0 > DEADLINE) {
            __hip_atomic_store(tmo, call, __ATOMIC_RELAXED, __HIP_MEMORY_SCOPE_AGENT);
            return false;
        }
        __builtin_amdgcn_s_sleep(4);
    }
}

__device__ __forceinline__ void load_x_unit(u32x4 (&v)[8], rsrc_t rx, unsigned row0, unsigned k0, int lane) {
    const unsigned a7 = lane >> 3, kcx = (lane & 7) ^ (((a7 >> 1) & 1) << 2);
#pragma unroll
    for (int j = 0; j < 8; ++j) v[j] = ld16(rx, ((row0 + a7 * 8 + j) * 1024u + k0 + ((kcx ^ j) * 8)) * 2u);
}
template <bool PACED>
__device__ __forceinline__ void store_x_unit(const u32x4 (&v)[8], rsrc_t rd, unsigned off, int lane) {
#pragma unroll
    for (int j = 0; j < 8; ++j) { st16(v[j], rd, off + j * 1024 + lane * 16); if (PACED && j < 7) WAIT_V(0); }
}
__device__ __forceinline__ void load_w_unit(u32x4 (&v)[8], rsrc_t rw, unsigned col0, unsigned k0, int lane) {
    const unsigned a7 = lane & 7, kc = lane >> 3;
#pragma unroll
    for (int i = 0; i < 8; ++i) v[i] = ld16(rw, ((k0 + kc * 8 + i) * 8192u + col0 + a7 * 8) * 2u);
}
template <bool PACED>
__device__ __forceinline__ void store_w_unit(const u32x4 (&v)[8], rsrc_t rd, unsigned off, int lane) {
    const unsigned a7 = lane & 7, kc = (lane >> 3) ^ (((a7 >> 1) & 1) << 2);
#pragma unroll
    for (int j = 0; j < 8; ++j) {
        u32x4 o;
#pragma unroll
        for (int e = 0; e < 4; ++e) o[e] = __builtin_amdgcn_perm(v[2 * e + 1][j >> 1], v[2 * e][j >> 1], (j & 1) ? 0x07060302u : 0x05040100u);
        st16(o, rd, off + j * 1024 + a7 * 128 + ((kc ^ j) * 16));
        if (PACED && j < 7) WAIT_V(0);
    }
}

__device__ __forceinline__ unsigned cvt_pk_bf16(float lo, float hi) { unsigned r; asm volatile("v_cvt_pk_bf16_f32 %0, %1, %2" : "=v"(r) : "v"(lo), "v"(hi)); return r; }
__device__ __forceinline__ float silu(float y) {
    const float t = fminf(fmaxf(y, -60.0f), 60.0f);
    const float e = __builtin_amdgcn_exp2f(t * -1.44269504088896341f);
    return y * __builtin_amdgcn_rcpf(1.0f + e);
}

__device__ __forceinline__ unsigned own_off(unsigned o) { return ((o >> 2) * 4096u * 8192u + ((o >> 1) & 1) * 4096u + (o & 1) * 2048u) * 2u; }
__device__ __forceinline__ unsigned rj_off(unsigned rj) { return ((rj >> 3) * 256u * 8192u + (rj & 7) * 256u) * 2u; }

__global__ __launch_bounds__(512, 2) void gemm_ar_kernel(Params P) {
    extern __shared__ __attribute__((aligned(16))) unsigned char shm[];
    LAS unsigned char* lds = (LAS unsigned char*)shm;
    const int tid = threadIdx.x, lane = tid & 63;
    const int wid = __builtin_amdgcn_readfirstlane(tid >> 6);
    const unsigned rank = (unsigned)P.rank & 7u;
    char* myws = P.ws[rank];
    unsigned* myfl = (unsigned*)(myws + OFF_FL);
    unsigned* tmo = myfl + F_TMO / 4;
    LAS unsigned* lw = (LAS unsigned*)(lds + 131072);
    const unsigned bid = blockIdx.x;
    const unsigned qi = rank >> 2, qj = (rank >> 1) & 1, hh = rank & 1;
    const rsrc_t rx = mkrs(P.x, X_BYTES), rw = mkrs(P.w, W_BYTES), rmy = mkrs(myws, WS_USED);

    unsigned call = 0;
    if (tid == 0) call = __hip_atomic_fetch_add(myfl + F_CTR / 4, 1u, __ATOMIC_RELAXED, __HIP_MEMORY_SCOPE_AGENT) / (unsigned)NWG + 1u;
    STAMP(0);

    auto self_load = [&](u32x4 (&v)[8], unsigned su) {
        const unsigned b = su >> 6, step = b >> 1, u = su & 63;
        if (!(b & 1)) load_x_unit(v, rx, qi * 4096u + step * 256u + (u & 3) * 64u, (u >> 2) * 64u, lane);
        else { const unsigned co = (unsigned)(kColPacked >> (4 * step)) & 15u, c = (((co >> 3) ^ hh ^ 1) & 1) * 8 + (co & 7);
               load_w_unit(v, rw, qj * 4096u + c * 256u + (u & 3) * 64u, (u >> 2) * 64u, lane); }
    };
    auto self_store = [&](const u32x4 (&v)[8], unsigned su) -> unsigned* {
        const unsigned b = su >> 6, step = b >> 1, u = su & 63, fw = (u >> 1) * 8 + 6 + (u & 1);
        if (!(b & 1)) { store_x_unit<false>(v, rmy, OFF_XS + step * BLK_BYTES + u * UNIT_BYTES, lane); return myfl + (F_BLK + step * 1024) / 4 + fw; }
        const unsigned co = (unsigned)(kColPacked >> (4 * step)) & 15u, c = (((co >> 3) ^ hh ^ 1) & 1) * 8 + (co & 7);
        store_w_unit<false>(v, rmy, OFF_WS + c * BLK_BYTES + u * UNIT_BYTES, lane);
        return myfl + (F_BLK + (16 + c) * 1024) / 4 + fw;
    };

    if (bid < NCOMM) {
        const unsigned e = bid / WPL + 1, wg = bid % WPL;
        const unsigned dst = (rank ^ e) & 7u;
        char* dws = P.ws[dst];
        unsigned* dfl = (unsigned*)(dws + OFF_FL);
        const rsrc_t rdw = mkrs(dws, WS_USED), rdo = mkrs(P.out[dst], OUT_BYTES), rmo = mkrs(P.out[rank], OUT_BYTES);
        const unsigned dqi = dst >> 2, dqj = (dst >> 1) & 1, dh = dst & 1;
        const unsigned rsrc_rank = dst ^ 1u, rho = (e ^ 1u) - 2u;
        const unsigned nrel = e == 1 ? 0u : (NREL - rho + 5) / 6;
        const unsigned n0 = e == 1 ? (256 - NGEMM) * 8 : 1024u;
        const unsigned total = n0 + (e == 1 ? (128 + NDIRECT) : (128 + nrel)) * TU;
        unsigned* queue = myfl + (F_Q + e * 128) / 4;
        const unsigned trj = kTileRJ[lane] | ((unsigned)kTileRJ[lane + 64] << 8);
        auto out_tile_off = [&](unsigned o, unsigned i) { const unsigned t = (unsigned)__builtin_amdgcn_readlane((int)trj, (int)(i & 63)); return own_off(o) + rj_off((t >> ((i >> 6) * 8)) & 255u); };
        unsigned* pend = nullptr;
        unsigned long long m0 = 0, m1 = 0; unsigned mkind = 0;
        bool ok = true, first = true;
        if (tid == 0) lw[1] = 0;
        unsigned t = wg * 8 + wid;
        while (t < total) {
            unsigned kind, idx = 0, ch = 0;
            if (t < n0) kind = e == 1 ? 0u : 1u + ((t >> 5) & 1u);
            else {
                const unsigned n = (t - n0) / TU; ch = (t - n0) % TU;
                unsigned dep = 0;
                if (e == 1) { if (n < 128) { kind = 3; idx = n; } else { kind = 4; idx = n - 128 + NREL; } }
                else {
                    kind = 4; idx = n;
                    if (n >= rho + RDELAY + 1) { const unsigned a = n - (rho + RDELAY + 1), m = a / 7, rem = a % 7;
                        if (m >= nrel) idx = n - nrel; else if (rem == 0) { kind = 5; idx = rho + 6 * m; dep = m * TU + ch; } else idx = n - (m + 1); }
                }
                if (ok && (kind != mkind || !(((idx < 64 ? m0 : m1) >> (idx & 63)) & 1ull))) {
                    WAIT_V(0);
                    if (pend && lane == 0) st_flag(pend, call);
                    pend = nullptr;
                    if (kind == 5) ok = wave_wait(myfl + F_FR / 4 + rsrc_rank * 256 + dep, true, call, tmo);
                    else {
                        const unsigned* fp = myfl + (kind == 4 ? F_LF : F_LP) / 4;
                        ok = wave_wait(fp + idx, true, call, tmo);
                        const unsigned f0 = ld_flag(fp + lane), f1 = ld_flag(fp + 64 + lane);
                        m0 = __ballot(f0 >= call); m1 = __ballot(f1 >= call); mkind = kind;
                    }
                }
            }
            unsigned nt = 0;
            if (lane == 0) nt = __hip_atomic_fetch_add(queue, 1u, __ATOMIC_RELAXED, __HIP_MEMORY_SCOPE_AGENT);
            u32x4 v[8];
            unsigned o = 0;
            const unsigned step = (t >> 6) & 15u, u = t & 31;
            const unsigned co = (unsigned)(kColPacked >> (4 * step)) & 15u, c = (((co >> 3) ^ dh ^ 1) & 1) * 8 + (co & 7);
            if (ok) {
                if (kind == 0) self_load(v, NGEMM * 8 + t);
                else if (kind == 1) load_x_unit(v, rx, dqi * 4096u + step * 256u + (u & 3) * 64u, dh * 512u + (u >> 2) * 64u, lane);
                else if (kind == 2) load_w_unit(v, rw, dqj * 4096u + c * 256u + (u & 3) * 64u, dh * 512u + (u >> 2) * 64u, lane);
                else if (kind == 3) {
                    o = idx * TILE_BYTES + ch * UNIT_BYTES + lane * 16;
#pragma unroll
                    for (int i = 0; i < 8; ++i) v[i] = ld16c(rmy, OFF_PL + o + i * 1024);
                } else {
                    o = out_tile_off(kind == 4 ? rank : rsrc_rank, idx) + (ch * 16 + (lane >> 5)) * 16384u + (lane & 31) * 16;
#pragma unroll
                    for (int i = 0; i < 8; ++i) v[i] = ld16c(rmo, o + i * 32768u);
                }
            }
            WAIT_V(0);
            if (first) { if (tid == 0) lw[0] = call; wg_barrier(); call = rfl(lw[0]); first = false; }
            if (pend && lane == 0) st_flag(pend, call);
            pend = nullptr;
            if (ok) {
                const unsigned uoff = (16 + (e - 2) * 8) * KT_BYTES + u * UNIT_BYTES, fw = u * 8 + (e - 2);
                if (kind == 0) pend = self_store(v, NGEMM * 8 + t);
                else if (kind == 1) { store_x_unit<true>(v, rdw, OFF_XS + step * BLK_BYTES + uoff, lane); pend = dfl + (F_BLK + step * 1024) / 4 + fw; }
                else if (kind == 2) { store_w_unit<true>(v, rdw, OFF_WS + c * BLK_BYTES + uoff, lane); pend = dfl + (F_BLK + (16 + c) * 1024) / 4 + fw; }
                else if (kind == 3) {
#pragma unroll
                    for (int i = 0; i < 8; ++i) { st16(v[i], rdw, OFF_PB + o + i * 1024); if (i < 7) WAIT_V(0); }
                    pend = dfl + (F_FP + idx * 128) / 4 + ch;
                } else {
#pragma unroll
                    for (int i = 0; i < 8; ++i) { st16(v[i], rdo, o + i * 32768u); if (i < 7) WAIT_V(0); }
                    if (kind == 4 && e != 1 && idx < NREL && idx % 6 == e - 2) pend = dfl + F_FR / 4 + rank * 256 + (idx / 6) * TU + ch;
                }
            }
            t = rfl(nt) - (call - 1u) * total + WPL * 8;
        }
        WAIT_V(0);
        if (lane == 0 && ok) {
            if (pend) st_flag(pend, call);
            if (__hip_atomic_fetch_add(lw + 1, 1u, __ATOMIC_RELAXED, __HIP_MEMORY_SCOPE_WORKGROUP) == 7u) st_flag(dfl + F_FD / 4 + rank * 8 + wg, call);
        }
        STAMP(4);
        return;
    }

    const unsigned g = bid - NCOMM;
    {
        u32x4 v[8];
        self_load(v, g * 8 + wid);
        unsigned* f = self_store(v, g * 8 + wid);
        if (tid == 0) lw[0] = call;
        WAIT_V(0); wg_barrier();
        call = rfl(lw[0]);
        if (lane == 0) st_flag(f, call);
    }
    STAMP(1);
    const int wr = wid >> 2, wc = wid & 3, fr = lane & 15, fq = lane >> 4;
    const unsigned voff = tid * 16;
    const unsigned ldsw = (unsigned)wid * 1024u;
    int aoff[2], boff[2];
#pragma unroll
    for (int q = 0; q < 2; ++q) {
        aoff[q] = wr * 8192 + (fr & 7) * 1024 + (fr >> 3) * 128 + (((fq ^ (fr & 7)) ^ (q << 2)) * 16);
        boff[q] = (wc >> 1) * 8192 + (fr & 3) * 1024 + (4 * (wc & 1) + (fr >> 2)) * 128 + ((fq ^ (fr & 3) ^ ((((fr >> 3) & 1) ^ q) << 2)) * 16);
    }
#define SA(b, h) (((b) * 2 + (h)) * HT_BYTES)
#define SB(b, h) ((4 + (b) * 2 + (h)) * HT_BYTES)
#define STAGE(bufoff, gbase) do { _Pragma("unroll") for (int _i = 0; _i < 2; ++_i) \
        __builtin_amdgcn_global_load_lds((const unsigned*)((const char*)(gbase) + voff + _i * 8192), (LAS unsigned*)(lds + (bufoff) + ldsw + _i * 8192), 16, 0, 0); } while (0)
#define LDA(dst, b, h) do { _Pragma("unroll") for (int m = 0; m < 4; ++m) _Pragma("unroll") for (int k = 0; k < 2; ++k) dst[m][k] = *(const LAS bf16x8*)(lds + SA(b, h) + aoff[(k ^ m) & 1] + m * 256); } while (0)
#define LDB(dst, b, h) do { _Pragma("unroll") for (int n = 0; n < 2; ++n) _Pragma("unroll") for (int k = 0; k < 2; ++k) dst[n][k] = *(const LAS bf16x8*)(lds + SB(b, h) + boff[(k ^ n) & 1] + n * 4096); } while (0)
#define MMA(ai, bj, At, Bt) do { __builtin_amdgcn_s_setprio(1); _Pragma("unroll") for (int m = 0; m < 4; ++m) _Pragma("unroll") for (int n = 0; n < 2; ++n) _Pragma("unroll") for (int k = 0; k < 2; ++k) \
        acc[ai][bj][m][n] = __builtin_amdgcn_mfma_f32_16x16x32_bf16(Bt[n][k], At[m][k], acc[ai][bj][m][n], 0, 0, 0); __builtin_amdgcn_s_setprio(0); } while (0)

    for (int round = 0; round < 2; ++round) {
        const unsigned tt = round == 0 ? kTile1[g] : kTile2[g];
        if (tt > 255u) break;
        const unsigned own = tt >> 7, ti = tt & 127;
        const unsigned rj = kTileRJ[ti], r = rj >> 3, c = ((own ^ hh ^ 1) & 1) * 8 + (rj & 7);
        if (wid == 0) {
            const unsigned* fp = myfl + (F_BLK + (lane < 32 ? r : 16 + c) * 1024) / 4 + (lane & 31);
            (void)wave_wait(fp + 7 * 32, true, call, tmo);
            (void)wave_wait_n<7>(fp, call, tmo);
            __builtin_amdgcn_fence(__ATOMIC_ACQUIRE, "");
            WAIT_V(0);
        }
        wg_barrier();
        STAMP(2 + 4 * round);
        f32x4 acc[2][2][4][2];
#pragma unroll
        for (int a = 0; a < 2; ++a)
#pragma unroll
            for (int b = 0; b < 2; ++b)
#pragma unroll
                for (int m = 0; m < 4; ++m)
#pragma unroll
                    for (int n = 0; n < 2; ++n) acc[a][b][m][n] = (f32x4){0.f, 0.f, 0.f, 0.f};
        bf16x8 At[4][2], B0[2][2], B1[2][2];
        const char* cA = myws + OFF_XS + r * BLK_BYTES;
        const char* cB = myws + OFF_WS + c * BLK_BYTES;
        constexpr int nt = 64;
        STAGE(SB(0, 0), cB); STAGE(SA(0, 0), cA); STAGE(SB(0, 1), cB + HT_BYTES); STAGE(SA(0, 1), cA + HT_BYTES);
        if (wr == 1) BAR;
        WAIT_V(4); BAR;
        STAGE(SB(1, 0), cB + KT_BYTES); STAGE(SA(1, 0), cA + KT_BYTES); STAGE(SB(1, 1), cB + HT_BYTES + KT_BYTES);
        WAIT_V(6); BAR;
        for (int t = 0; t < nt; t += 2) {
            const bool last = (t == nt - 2);
            const char* a1 = cA + (size_t)(t + 1) * KT_BYTES;
            const char* a2 = last ? cA : cA + (size_t)(t + 2) * KT_BYTES; const char* b2 = last ? cB : cB + (size_t)(t + 2) * KT_BYTES;
            const char* a3 = a2 + KT_BYTES; const char* b3 = b2 + KT_BYTES;
            LDB(B0, 0, 0); SCHED; LDA(At, 0, 0); STAGE(SA(1, 1), a1 + HT_BYTES);
            WAIT_L(8); BAR; WAIT_L(0); MMA(0, 0, At, B0); BAR; SCHED;
            LDB(B1, 0, 1); STAGE(SB(0, 0), b2);
            BAR; WAIT_L(0); MMA(0, 1, At, B1); BAR;
            LDA(At, 0, 1); STAGE(SA(0, 0), a2);
            BAR; WAIT_L(0); MMA(1, 0, At, B0); BAR; SCHED;
            STAGE(SB(0, 1), b2 + HT_BYTES);
            WAIT_V(6); BAR; MMA(1, 1, At, B1); BAR;
            LDB(B0, 1, 0); SCHED; LDA(At, 1, 0); STAGE(SA(0, 1), a2 + HT_BYTES);
            WAIT_L(8); BAR; WAIT_L(0); MMA(0, 0, At, B0); BAR; SCHED;
            LDB(B1, 1, 1); STAGE(SB(1, 0), b3);
            BAR; WAIT_L(0); MMA(0, 1, At, B1); BAR;
            LDA(At, 1, 1); STAGE(SA(1, 0), a3);
            BAR; WAIT_L(0); MMA(1, 0, At, B0); BAR; SCHED;
            STAGE(SB(1, 1), b3 + HT_BYTES);
            WAIT_V(6); BAR; MMA(1, 1, At, B1); BAR;
        }
        WAIT_V(0);
        if (wr == 0) BAR;
        BAR;
        STAMP(3 + 4 * round);
        if (!own) {
            const bool bad = rfl(__hip_atomic_load(tmo, __ATOMIC_RELAXED, __HIP_MEMORY_SCOPE_AGENT)) == call;
            const unsigned base = OFF_PL + ti * TILE_BYTES + tid * 16;
#pragma unroll
            for (int ai = 0; ai < 2; ++ai)
#pragma unroll
                for (int m = 0; m < 4; ++m)
#pragma unroll
                    for (int bj = 0; bj < 2; ++bj) {
                        const f32x4 v0 = acc[ai][bj][m][0], v1 = acc[ai][bj][m][1];
                        u32x4 o; o.x = cvt_pk_bf16(v0[0], v0[1]); o.y = cvt_pk_bf16(v0[2], v0[3]); o.z = cvt_pk_bf16(v1[0], v1[1]); o.w = cvt_pk_bf16(v1[2], v1[3]);
                        if (bad) o = (u32x4){0x7fc07fc0u, 0x7fc07fc0u, 0x7fc07fc0u, 0x7fc07fc0u};
                        st16(o, rmy, base + ((ai * 4 + m) * 2 + bj) * 8192);
                    }
            WAIT_V(0); wg_barrier();
            if (tid == 0) st_flag(myfl + F_LP / 4 + ti, call);
        } else {
            if (wid == 0) {
                (void)wave_wait(myfl + (F_FP + ti * 128) / 4 + (lane % TU), true, call, tmo);
                __builtin_amdgcn_fence(__ATOMIC_ACQUIRE, "");
                WAIT_V(0);
            }
            wg_barrier();
            STAMP(4 + 4 * round);
            const bool bad = rfl(__hip_atomic_load(tmo, __ATOMIC_RELAXED, __HIP_MEMORY_SCOPE_AGENT)) == call;
            const rsrc_t rmo = mkrs(P.out[rank], OUT_BYTES);
            const unsigned pbase = OFF_PB + ti * TILE_BYTES + tid * 16;
            const unsigned obase = own_off(rank) + rj_off(rj) + (wr * 64 + fr) * 16384u + (wc * 32 + fq * 8) * 2;
#pragma unroll
            for (int ai = 0; ai < 2; ++ai) {
                u32x4 pv[4][2];
#pragma unroll
                for (int m = 0; m < 4; ++m)
#pragma unroll
                    for (int bj = 0; bj < 2; ++bj) pv[m][bj] = bad ? (u32x4){0u, 0u, 0u, 0u} : ld16c(rmy, pbase + ((ai * 4 + m) * 2 + bj) * 8192);
#pragma unroll
                for (int m = 0; m < 4; ++m)
#pragma unroll
                    for (int bj = 0; bj < 2; ++bj) {
                        const f32x4 v0 = acc[ai][bj][m][0], v1 = acc[ai][bj][m][1]; const u32x4 p = pv[m][bj];
                        u32x4 o;
                        o.x = cvt_pk_bf16(silu(v0[0] + __uint_as_float(p.x << 16)), silu(v0[1] + __uint_as_float(p.x & 0xffff0000u)));
                        o.y = cvt_pk_bf16(silu(v0[2] + __uint_as_float(p.y << 16)), silu(v0[3] + __uint_as_float(p.y & 0xffff0000u)));
                        o.z = cvt_pk_bf16(silu(v1[0] + __uint_as_float(p.z << 16)), silu(v1[1] + __uint_as_float(p.z & 0xffff0000u)));
                        o.w = cvt_pk_bf16(silu(v1[2] + __uint_as_float(p.w << 16)), silu(v1[3] + __uint_as_float(p.w & 0xffff0000u)));
                        if (bad) o = (u32x4){0x7fc07fc0u, 0x7fc07fc0u, 0x7fc07fc0u, 0x7fc07fc0u};
                        st16(o, rmo, obase + (ai * 128 + m * 16) * 16384u + bj * 256);
                    }
            }
            WAIT_V(0); wg_barrier();
            if (tid == 0) st_flag(myfl + F_LF / 4 + ti, call);
            STAMP(5 + 4 * round);
            if (ti == 0 && wid == 0) {
                const bool use = (unsigned)(lane >> 3) != rank;
                const bool ok = wave_wait(myfl + F_FD / 4 + lane, use, call, tmo);
                __builtin_amdgcn_fence(__ATOMIC_ACQUIRE, "");
                if (!ok && lane == 0) __builtin_amdgcn_raw_buffer_store_b16((short)0x7fc0, rmo, own_off(rank) + rj_off(rj), 0, AUX_WT);
                STAMP(10);
            }
        }
    }
}

}

extern "C" void kernel_launch(int rank, int nranks, ncclComm_t comm,
                              hipStream_t stream,
                              void* const* d_in, const int* in_sizes, int n_in,
                              void* d_out, int out_size,
                              const int* in_dtypes, int out_dtype,
                              void* const* d_out_all,
                              void* const* d_ws_all, size_t ws_bytes) {
    if (nranks != NR || (unsigned)rank >= (unsigned)NR || ws_bytes < (size_t)WS_USED || n_in < 2) {
        fprintf(stderr, "unsupported configuration\n");
        return;
    }
    Params P;
    P.x = (const char*)d_in[0]; P.w = (const char*)d_in[1]; P.rank = rank;
    for (int p = 0; p < NR; ++p) { P.out[p] = (char*)d_out_all[p]; P.ws[p] = (char*)d_ws_all[p]; }
    HIP_CHECK(hipFuncSetAttribute((const void*)gemm_ar_kernel, hipFuncAttributeMaxDynamicSharedMemorySize, LDS_BYTES));
    hipLaunchKernelGGL(gemm_ar_kernel, dim3(NWG), dim3(512), LDS_BYTES, stream, P);
    HIP_CHECK(hipGetLastError());
}
```

```cpp
#include <hip/hip_runtime.h>
#include <rccl/rccl.h>
#include <cstdio>

#define HIP_CHECK(expr) do { hipError_t _e = (expr); if (_e != hipSuccess) { \
    fprintf(stderr, "HIP error %s at %s:%d\n", hipGetErrorString(_e), __FILE__, __LINE__); \
    return; } } while (0)

namespace {

#define LAS __attribute__((address_space(3)))
typedef short bf16x8 __attribute__((ext_vector_type(8)));
typedef float f32x4 __attribute__((ext_vector_type(4)));
typedef unsigned u32x4 __attribute__((ext_vector_type(4)));
typedef __amdgpu_buffer_rsrc_t rsrc_t;

constexpr int NR = 8;
constexpr unsigned MiB = 1u << 20;
constexpr unsigned X_BYTES = 8192u * 1024u * 2u, W_BYTES = 1024u * 8192u * 2u, OUT_BYTES = 8192u * 8192u * 2u;
constexpr unsigned OFF_XS = 0, OFF_WS = 32 * MiB;
constexpr unsigned OFF_PB = 64 * MiB, OFF_PL = 80 * MiB;
constexpr unsigned OFF_EX = 96 * MiB, OFF_EW = 112 * MiB;
constexpr unsigned OFF_FT = 128 * MiB;
constexpr unsigned OFF_FL = 144 * MiB;
constexpr unsigned OFF_SL = 146 * MiB, WS_USED = 274 * MiB;
constexpr unsigned BLK_BYTES = 2 * MiB, EBLK_BYTES = MiB / 2, KT_BYTES = 32768, HT_BYTES = 16384, UNIT_BYTES = 8192, TILE_BYTES = 131072;
constexpr unsigned F_CTR = 0, F_TMO = 128, F_RP = 2048, F_OWN = 4096  , F_FP = 8192  ,
                   F_PG = 24576  , F_EXF = 28672  , F_BLK = 32768  , F_RF = 65536  , F_SL = 69632  ;
constexpr unsigned F_DBG = 262144;
constexpr unsigned F_QS = 1048576, QSTRIDE = 0x11100;
constexpr int WPL = 8;
#define WPW 7
#define EXD 200
constexpr int NCOMM = 7 * WPL, NGEMM = 200, NWG = NCOMM + NGEMM;
constexpr unsigned TU = 16;
constexpr unsigned NFLOAT = 128 * TU;
static_assert(WPL * WPW <= 64, "progress words");
#define DEADLINE_TICKS 100000000ull
constexpr unsigned long long DEADLINE = DEADLINE_TICKS;
constexpr int LDS_BYTES = 131072 + 2048;
constexpr int AUX_WT = 17;

static constexpr unsigned char kColOrder[16] = {0,1,2,8,3,9,10,4,5,6,11,7,12,13,14,15};
static constexpr unsigned char kTileRJ[128] = {0,8,1,9,16,17,2,10,18,24,25,26,32,33,34,3,11,19,27,35,40,41,42,43,48,49,50,51,56,57,58,59,4,64,65,66,67,12,20,28,36,44,52,60,68,5,13,21,29,37,45,72,73,74,75,80,81,82,83,88,89,90,91,96,97,98,99,76,84,92,100,104,105,106,107,108,53,61,69,77,85,93,101,109,112,113,114,115,116,117,6,14,22,30,38,46,54,62,70,78,86,94,102,110,118,120,121,122,123,124,125,126,7,15,23,31,39,47,55,63,71,79,87,95,103,111,119,127};
static constexpr unsigned char kFinRJ[128] = {0,8,16,24,32,40,1,9,17,25,33,41,48,49,2,10,18,26,34,42,50,56,57,58,64,65,66,3,11,19,27,35,43,51,59,67,72,73,74,75,80,81,82,83,4,12,20,28,36,44,52,60,68,88,89,90,91,5,13,21,29,37,45,96,97,98,99,76,84,92,100,104,105,106,107,108,53,61,69,77,85,93,101,109,112,113,114,115,116,117,6,14,22,30,38,46,54,62,70,78,86,94,102,110,118,120,121,122,123,124,125,126,7,15,23,31,39,47,55,63,71,79,87,95,103,111,119,127};
static constexpr unsigned char kP2F[128] = {0,1,6,7,2,8,14,15,16,3,9,17,4,10,18,27,28,29,30,31,5,11,19,32,12,13,20,33,21,22,23,34,44,24,25,26,35,45,46,47,48,49,50,51,52,57,58,59,60,61,62,36,37,38,39,40,41,42,43,53,54,55,56,63,64,65,66,67,68,69,70,71,72,73,74,75,76,77,78,79,80,81,82,83,84,85,86,87,88,89,90,91,92,93,94,95,96,97,98,99,100,101,102,103,104,105,106,107,108,109,110,111,112,113,114,115,116,117,118,119,120,121,122,123,124,125,126,127};
static constexpr unsigned kTask[800] = {131200,4294967295,4294967295,4294967295,131213,4294967295,4294967295,4294967295,131235,4294967295,4294967295,4294967295,33426947,142478864,298716199,131164,14696449,106971148,255885342,131151,57197574,206145560,131174,4294967295,158977042,131243,4294967295,4294967295,15533569,107808268,256773150,131275,131201,4294967295,4294967295,4294967295,131221,4294967295,4294967295,4294967295,82642441,131126,4294967295,4294967295,40141316,172295188,398491695,131165,41257476,180683797,398787631,131167,131266,4294967295,4294967295,4294967295,159256082,131244,4294967295,4294967295,23643138,132695055,273254432,131279,131204,4294967295,4294967295,4294967295,131225,4294967295,4294967295,4294967295,90193930,131252,4294967295,4294967295,40699396,172887060,399083567,131166,41815556,181275669,399379503,131168,165691411,131232,4294967295,4294967295,131248,4294967295,4294967295,4294967295,24201218,133253135,281051173,131280,131209,4294967295,4294967295,4294967295,131206,4294967295,4294967295,4294967295,90472970,131253,4294967295,4294967295,98582539,131215,4294967295,4294967295,48529925,189072406,131169,4294967295,165970451,131237,4294967295,4294967295,131249,4294967295,4294967295,4294967295,24759298,133811215,281643045,131281,131212,4294967295,4294967295,4294967295,131207,4294967295,4294967295,4294967295,149751313,131139,4294967295,4294967295,98861579,131216,4294967295,4294967295,49088005,189664278,131170,4294967295,166249491,131238,4294967295,4294967295,6307840,115359757,239108124,131250,31473667,140525584,289439782,131282,131220,4294967295,4294967295,4294967295,131208,4294967295,4294967295,4294967295,90752010,131130,4294967295,4294967295,99140619,131217,4294967295,4294967295,49646085,197461015,131171,4294967295,16649729,108924428,265161759,131155,16091649,108366348,264569887,131276,32031747,141083664,290031654,131283,82921481,131124,131327,4294967295,131211,4294967295,4294967295,4294967295,92147210,131256,4294967295,4294967295,99419659,131218,4294967295,4294967295,15254529,107529228,256477214,131152,57755654,213942297,131175,4294967295,16370689,108645388,264865823,131277,40420356,172591124,131288,4294967295,83200521,131125,4294967295,4294967295,131214,4294967295,4294967295,4294967295,124306446,131257,4294967295,4294967295,99698699,131219,4294967295,4294967295,50204165,198052887,131172,4294967295,14975489,107250188,256181278,131274,23085058,132136975,272662560,131278,40978436,180387861,131289,4294967295,83758601,131251,4294967295,4294967295,131222,4294967295,4294967295,4294967295,150309393,131140,4294967295,4294967295,75090952,315197481,131184,4294967295,99977739,131223,4294967295,4294967295,6586880,115638797,239404060,131245,48808965,189368342,131292,4294967295,58592774,214830105,131299,4294967295,91031050,131127,4294967295,4294967295,131143,4294967295,4294967295,4294967295,124864526,131134,4294967295,4294967295,340363297,131185,4294967295,4294967295,100256779,131227,4294967295,4294967295,6865920,115917837,239699996,131246,49367045,197165079,131293,4294967295,65307143,222626842,131300,4294967295,91310090,131128,4294967295,4294967295,131144,4294967295,4294967295,4294967295,149193233,131260,4294967295,4294967295,348160034,131186,4294967295,4294967295,100535819,131231,4294967295,4294967295,7144960,116196877,239995932,131247,49925125,197756951,131294,4294967295,65865223,223218714,131301,4294967295,91589130,131129,4294967295,4294967295,131145,4294967295,4294967295,4294967295,149472273,131261,4294967295,4294967295,348751906,131187,4294967295,4294967295,123748366,131236,4294967295,4294967295,32310787,141362704,290327590,131161,56639494,205553688,131295,4294967295,66423303,231015451,131302,4294967295,91868170,131255,4294967295,4294967295,7703040,116754957,247792669,131271,158139922,131141,4294967295,4294967295,356548643,131188,4294967295,4294967295,124027406,131254,4294967295,4294967295,58313734,214534169,131176,4294967295,56918534,205849624,131296,4294967295,66981383,231607323,131303,4294967295,125143566,131131,4294967295,4294967295,23364098,132416015,272958496,131156,150588433,131138,4294967295,4294967295,25038338,134090255,281938981,131159,124585486,131258,4294967295,4294967295,39862276,171999252,131287,4294967295,57476614,206441496,131297,4294967295,73695752,306512936,131304,4294967295,125422606,131132,4294967295,4294967295,23922178,132974095,273550368,131157,158418962,131264,4294967295,4294967295,32868867,141920784,298124327,131285,15812609,108087308,264273951,131153,41536516,180979733,131290,4294967295,58034694,214238233,131298,4294967295,348455970,131309,4294967295,4294967295,125701646,131133,4294967295,4294967295,24480258,133532175,281347109,131158,167644691,131265,4294967295,4294967295,33147907,142199824,298420263,131286,357140515,131189,4294967295,4294967295,48250885,188776470,131291,4294967295,365825060,131314,4294967295,4294967295,349047842,131310,4294967295,4294967295,148914193,131259,4294967295,4294967295,32589827,141641744,297828391,131284,158698002,131142,4294967295,4294967295,31752707,140804624,289735718,131160,364937252,131190,4294967295,4294967295,73974792,306808872,131182,4294967295,373621804,131315,4294967295,4294967295,356844579,131311,4294967295,4294967295,150867473,131135,4294967295,4294967295,65028103,222330906,131177,4294967295,131146,4294967295,4294967295,4294967295,66702343,231311387,131180,4294967295,365529124,131191,4294967295,4294967295,74532872,314605609,131183,4294967295,374213676,131316,4294967295,4294967295,390103086,131320,4294967295,4294967295,157302802,131136,4294967295,4294967295,65586183,222922778,131178,4294967295,7982080,117033997,248088605,131272,74811912,314901545,131306,4294967295,373325868,131192,4294967295,4294967295,340659233,131308,4294967295,4294967295,382010413,131317,4294967295,4294967295,390399022,131321,4294967295,4294967295,157581842,131137,4294967295,4294967295,66144263,230719515,131179,4294967295,8261120,117313037,248384541,131273,75369992,315493417,131307,4294967295,373917740,131193,4294967295,4294967295,357436451,131312,4294967295,4294967295,382306349,131318,4294967295,4294967295,390694958,131322,4294967295,4294967295,157860882,131263,4294967295,4294967295,74253832,307104808,131305,4294967295,7424000,116475917,247496733,131147,73416712,306217000,131181,4294967295,381714477,131194,4294967295,4294967295,365233188,131313,4294967295,4294967295,382602285,131319,4294967295,4294967295,390990894,131323,4294967295,4294967295,131224,4294967295,4294967295,4294967295,131202,4294967295,4294967295,4294967295,131226,4294967295,4294967295,4294967295,81805321,131120,131196,4294967295,82084361,131122,131198,4294967295,339771425,131173,4294967295,4294967295,166528531,131239,4294967295,4294967295,131267,4294967295,4294967295,4294967295,131228,4294967295,4294967295,4294967295,131203,4294967295,4294967295,4294967295,131229,4294967295,4294967295,4294967295,82363401,131121,131197,4294967295,323586090,131148,131199,4294967295,150030353,131262,4294967295,4294967295,166807571,131240,4294967295,4294967295,131268,4294967295,4294967295,4294967295,131233,4294967295,4294967295,4294967295,131205,4294967295,4294967295,4294967295,131230,4294967295,4294967295,4294967295,331974699,131162,4294967295,4294967295,323882026,131149,131324,4294967295,340067361,131195,4294967295,4294967295,167086611,131241,4294967295,4294967295,322994218,131269,4294967295,4294967295,83479561,131123,131326,4294967295,131210,4294967295,4294967295,4294967295,131234,4294967295,4294967295,4294967295,332270635,131163,4294967295,4294967295,331382827,131150,131325,4294967295,331678763,131154,4294967295,4294967295,167365651,131242,4294967295,4294967295,323290154,131270,4294967295,4294967295};
static constexpr unsigned char kExp[200] = {255,71,77,255,255,255,62,255,255,40,46,255,255,60,94,255,255,72,78,255,255,92,63,255,255,41,47,86,255,61,95,255,255,73,79,55,255,93,255,255,255,42,48,87,255,255,255,255,32,74,80,56,255,255,255,255,64,43,49,88,255,255,255,255,33,75,81,255,57,255,255,255,65,44,50,255,89,255,255,255,34,76,82,255,58,255,255,255,66,45,51,255,90,255,255,255,35,255,83,255,59,255,255,255,67,255,52,255,91,255,255,255,36,255,84,255,255,255,255,255,68,255,53,255,255,255,255,255,37,255,85,255,255,255,255,255,69,255,54,255,255,255,255,255,38,255,255,255,255,255,255,255,70,255,255,255,255,255,255,255,39,255,255,255,255,255,255,255,0,1,2,3,4,5,6,7,8,9,10,11,12,13,14,15,16,17,18,19,20,21,22,23,24,25,26,27,28,29,30,31};
constexpr unsigned long long pack_col_order() { unsigned long long v = 0; for (int i = 0; i < 16; ++i) v |= (unsigned long long)kColOrder[i] << (4 * i); return v; }
constexpr unsigned long long kColPacked = pack_col_order();

struct Params {
    const char* x; const char* w;
    char* out[NR]; char* ws[NR];
    int rank;
};

__device__ __forceinline__ rsrc_t mkrs(const void* p, unsigned n) { return __builtin_amdgcn_make_buffer_rsrc((void*)p, 0, n, 0x00020000); }
__device__ __forceinline__ u32x4 ld16(rsrc_t r, unsigned off) { return __builtin_amdgcn_raw_buffer_load_b128(r, off, 0, 0); }
__device__ __forceinline__ u32x4 ld16c(rsrc_t r, unsigned off, unsigned soff = 0) { return __builtin_amdgcn_raw_buffer_load_b128(r, off, soff, AUX_WT); }
__device__ __forceinline__ void st16(u32x4 v, rsrc_t r, unsigned off, unsigned soff = 0) { __builtin_amdgcn_raw_buffer_store_b128(v, r, off, soff, AUX_WT); }
#define WAIT_V(n) asm volatile("s_waitcnt vmcnt(" #n ")" ::: "memory")
#define WAIT_B(n) do { asm volatile("" ::: "memory"); __builtin_amdgcn_s_waitcnt(0x0F70 | (n)); asm volatile("" ::: "memory"); } while (0)
#define WAIT_L(n) asm volatile("s_waitcnt lgkmcnt(" #n ")" ::: "memory")
#define BAR __builtin_amdgcn_s_barrier()
#define SCHED __builtin_amdgcn_sched_barrier(0)
__device__ __forceinline__ void wg_barrier() { asm volatile("s_waitcnt lgkmcnt(0)" ::: "memory"); __builtin_amdgcn_s_barrier(); asm volatile("" ::: "memory"); }
__device__ __forceinline__ unsigned rfl(unsigned v) { return (unsigned)__builtin_amdgcn_readfirstlane((int)v); }
__device__ __forceinline__ unsigned ld_flag(const unsigned* p) { return __hip_atomic_load(p, __ATOMIC_RELAXED, __HIP_MEMORY_SCOPE_SYSTEM); }
__device__ __forceinline__ void st_flag(unsigned* p, unsigned v) { __hip_atomic_store(p, v, __ATOMIC_RELAXED, __HIP_MEMORY_SCOPE_SYSTEM); }
typedef __attribute__((address_space(1))) unsigned gu32;
__device__ __forceinline__ unsigned draw(unsigned* p) { gu32* q = (gu32*)p; asm volatile("" : "+v"(q)); return __hip_atomic_fetch_add(q, 1u, __ATOMIC_RELAXED, __HIP_MEMORY_SCOPE_AGENT); }

#define STAMP(slot) do {} while (0)
#define STAMPM(k) do {} while (0)
#define STAMPT(ti) do {} while (0)

template <int NW>
__device__ __forceinline__ bool wave_wait(const unsigned* p, bool use, unsigned want, unsigned call, unsigned* tmo) {
    const unsigned long long t0 = __builtin_amdgcn_s_memrealtime();
    for (;;) {
        bool all = true;
        const unsigned tm = __hip_atomic_load(tmo, __ATOMIC_RELAXED, __HIP_MEMORY_SCOPE_AGENT);
        if (use) {
            unsigned v[NW];
#pragma unroll
            for (int k = 0; k < NW; ++k) v[k] = ld_flag(p + k * 32);
#pragma unroll
            for (int k = 0; k < NW; ++k) all = all && v[k] >= want;
        }
        if (__all(all)) return true;
        if (rfl(tm) == call || __builtin_amdgcn_s_memrealtime() - t0 > DEADLINE) {
            __hip_atomic_store(tmo, call, __ATOMIC_RELAXED, __HIP_MEMORY_SCOPE_AGENT);
            return false;
        }
        __builtin_amdgcn_s_sleep(4);
    }
}

__device__ __forceinline__ void load_x_unit(u32x4 (&v)[8], rsrc_t rx, unsigned row0, unsigned k0, int lane) {
    const unsigned a7 = lane >> 3, kcx = (lane & 7) ^ (((a7 >> 1) & 1) << 2);
#pragma unroll
    for (int j = 0; j < 8; ++j) v[j] = ld16(rx, ((row0 + a7 * 8 + j) * 1024u + k0 + ((kcx ^ j) * 8)) * 2u);
}
__device__ __forceinline__ void load_w_unit(u32x4 (&o)[8], rsrc_t rw, unsigned col0, unsigned k0, int lane) {
    const unsigned a7 = lane & 7, kc0 = lane >> 3;
    u32x4 v[8];
#pragma unroll
    for (int i = 0; i < 8; ++i) v[i] = ld16(rw, ((k0 + kc0 * 8 + i) * 8192u + col0 + a7 * 8) * 2u);
#pragma unroll
    for (int j = 0; j < 8; ++j)
#pragma unroll
        for (int e = 0; e < 4; ++e) o[j][e] = __builtin_amdgcn_perm(v[2 * e + 1][j >> 1], v[2 * e][j >> 1], (j & 1) ? 0x07060302u : 0x05040100u);
}

__device__ __forceinline__ unsigned cvt_pk_bf16(float lo, float hi) { unsigned r; asm volatile("v_cvt_pk_bf16_f32 %0, %1, %2" : "=v"(r) : "v"(lo), "v"(hi)); return r; }
__device__ __forceinline__ float silu(float y) {
    const float t = fminf(fmaxf(y, -60.0f), 60.0f);
    const float e = __builtin_amdgcn_exp2f(t * -1.44269504088896341f);
    return y * __builtin_amdgcn_rcpf(1.0f + e);
}

__device__ __forceinline__ unsigned own_off(unsigned o) { return ((o >> 2) * 4096u * 8192u + ((o >> 1) & 1) * 4096u + (o & 1) * 2048u) * 2u; }
__device__ __forceinline__ unsigned rj_off(unsigned rj) { return ((rj >> 3) * 256u * 8192u + (rj & 7) * 256u) * 2u; }
__device__ __forceinline__ unsigned col_of(unsigned s, unsigned h) { const unsigned co = (unsigned)(kColPacked >> (4 * s)) & 15u; return (((co >> 3) ^ h ^ 1) & 1) * 8 + (co & 7); }

__global__ __launch_bounds__(512, 2) void gemm_ar_kernel(Params P) {
    extern __shared__ __attribute__((aligned(16))) unsigned char shm[];
    LAS unsigned char* lds = (LAS unsigned char*)shm;
    const int tid = threadIdx.x, lane = tid & 63;
    const int wid = __builtin_amdgcn_readfirstlane(tid >> 6);
    const unsigned rank = (unsigned)P.rank & 7u;
    char* myws = P.ws[rank];
    unsigned* myfl = (unsigned*)(myws + OFF_FL);
    unsigned* tmo = myfl + F_TMO / 4;
    LAS unsigned* lw = (LAS unsigned*)(lds + 131072);
    const unsigned bid = blockIdx.x;
    const unsigned qi = rank >> 2, qj = (rank >> 1) & 1, hh = rank & 1;
    const rsrc_t rmy = mkrs(myws, WS_USED);

    unsigned call = 0, tick = 0;
    STAMP(0);
    const unsigned long long tstart = __builtin_amdgcn_s_memrealtime();
    const unsigned xi = bid < NCOMM ? 255u : (bid >= NWG - 32 ? bid - (NWG - 32) : kExp[bid - NCOMM]);
    if (xi == 255u) {
        if (bid >= NCOMM) __builtin_amdgcn_s_sleep(64);
        if (tid == 0) lw[0] = draw(myfl + F_CTR / 4) / (unsigned)NWG + 1u;
        wg_barrier();
        call = rfl(lw[0]);
    } else {
        const rsrc_t rx = mkrs(P.x, X_BYTES), rw = mkrs(P.w, W_BYTES);
        for (unsigned step = xi >> 5; step < 16; step += 3) {
            const unsigned isw = (xi >> 4) & 1, q = (xi >> 3) & 1, h = (xi >> 2) & 1, kp = xi & 3;
            while (__builtin_amdgcn_s_memrealtime() - tstart < (unsigned long long)EXD * step) __builtin_amdgcn_s_sleep(4);
            if (step < 3 && tid == 0) tick = draw(myfl + F_CTR / 4);
            const unsigned u = (h * 4 + kp) * 8 + wid;
            const unsigned blk = isw ? col_of(step, h) : step;
            const bool push = step == 0 && !isw;
            u32x4 v[8];
            unsigned vo, kx;
            if (!isw) { load_x_unit(v, rx, q * 4096u + blk * 256u + (u & 3) * 64u, (u >> 2) * 64u, lane); vo = lane * 16; kx = 0; }
            else { load_w_unit(v, rw, q * 4096u + blk * 256u + (u & 3) * 64u, (u >> 2) * 64u, lane); vo = (lane & 7) * 128; kx = (lane >> 3) ^ ((((lane & 7) >> 1) & 1) << 2); }
            const unsigned xm = isw ? 7u : 0u;
            const unsigned d0 = isw ? q * 2 + h : q * 4 + h, d1 = d0 + (isw ? 4u : 2u);
            if (push) {
                const rsrc_t r0 = mkrs(P.ws[d0 & 7], (d0 >> 1) != (rank >> 1) ? WS_USED : 0u), r1 = mkrs(P.ws[d1 & 7], (d1 >> 1) != (rank >> 1) ? WS_USED : 0u);
                const unsigned ob = (isw ? OFF_WS + blk * BLK_BYTES : OFF_XS) + 16 * KT_BYTES + (kp * 8 + wid) * UNIT_BYTES + vo;
                const unsigned o0 = ob + (((rank ^ d0) - 2) & 7) * 8 * KT_BYTES, o1 = ob + (((rank ^ d1) - 2) & 7) * 8 * KT_BYTES;
#pragma unroll
                for (int j = 0; j < 8; ++j) { const unsigned z = j * 1024 + ((kx ^ (j & xm)) * 16); st16(v[j], r0, o0 + z); st16(v[j], r1, o1 + z); if (j < 7) WAIT_V(2); }
            }
#pragma unroll
            for (int j = 0; j < 8; ++j) st16(v[j], rmy, (isw ? OFF_EW : OFF_EX) + (q * 16 + blk) * EBLK_BYTES + u * UNIT_BYTES + vo + j * 1024 + ((kx ^ (j & xm)) * 16));
            if (step < 3) { asm volatile("" : "+v"(tick)); if (tid == 0) lw[0] = tick / (unsigned)NWG + 1u; }
            WAIT_V(0); wg_barrier();
            if (step < 3) call = rfl(lw[0]);
            if (push && lane < 2) {
                const unsigned d = lane ? d1 : d0;
                if ((d >> 1) != (rank >> 1)) st_flag((unsigned*)(P.ws[d & 7] + OFF_FL + F_BLK) + (isw ? (16 + blk) * 256 : 0u) + (kp * 8 + wid) * 8 + ((rank ^ d) - 2), call);
            }
            if (wid == 0 && lane < 3) {
                if (lane < 2) {
                    const unsigned d = lane ? d1 : d0;
                    if (!push && (d >> 1) != (rank >> 1)) st_flag((unsigned*)(P.ws[d & 7] + OFF_FL + F_EXF) + rank * 128 + step * 8 + isw * 4 + kp, call);
                } else if (q == (isw ? qj : qi)) st_flag(myfl + (F_OWN + (isw * 16 + blk) * 128) / 4 + h * 4 + kp, call);
            }
        }
    }
    STAMP(1);

    if (bid < NCOMM) {
        if (wid >= WPW) return;
        const unsigned e = bid / WPL + 1, wg = bid % WPL;
        const unsigned src = (rank ^ e) & 7u;
        char* sws = P.ws[src];
        char* sout = P.out[src];
        char* mout = P.out[rank];
        const unsigned n0 = e == 1 ? 0u : 1024u;
        const unsigned total = n0 + 128 * TU;
        unsigned* queue = myfl + (F_QS + e * QSTRIDE) / 4;
        unsigned* fqueue = myfl + F_QS / 4;
        unsigned* pgw = (unsigned*)(P.ws[src ^ 1u] + OFF_FL + F_PG) + rank * 64 + wg * WPW + wid;
        const unsigned trj = kFinRJ[lane] | ((unsigned)kFinRJ[lane + 64] << 8);
        auto group_of = [](unsigned n) { return n < 96 ? n >> 5 : (n < 112 ? 3u : (n < 120 ? 4u : (n < 124 ? 5u : 6u))); };
        unsigned posted = 0, topost = 0, pgmin = 0;
        unsigned long long m0 = 0, m1 = 0; unsigned mkind = 0;
        unsigned pend = 0;
        bool ok = true, flt = false;
        auto publish = [&]() {
            if (pend && lane == 0) st_flag(myfl + pend, call);
            pend = 0;
            if (topost > posted) { if (lane == 0 && ok) st_flag(pgw, call * 8u + topost); posted = topost; }
        };
        const unsigned skip = e != 1 ? 32u : 0u;
        const unsigned lbase = (call - 1u) * (total - skip) - WPL * WPW - skip, fbase = (call - 1u) * (NFLOAT + 7u * WPL * WPW);
        auto kind_of = [&](unsigned t, bool f) -> unsigned { return f ? (e == 1 ? 4u : 5u) : (t < n0 ? 1u : (e == 1 ? 3u : 4u)); };
        auto idx_of = [&](unsigned t, bool f, unsigned kind) -> unsigned { return kind == 1 ? t >> 3 : (f ? t : t - n0) / TU; };
        auto is_ready = [&](unsigned kind, unsigned idx) -> bool {
            if (!ok) return true;
            if (kind == 5) return pgmin > group_of(idx);
            return kind == mkind && (((idx < 64 ? m0 : m1) >> (idx & 63)) & 1ull);
        };
        auto flags_of = [&](unsigned kind) -> const unsigned* {
            return myfl + (kind == 1 ? F_EXF / 4 + src * 128 : (kind == 3 ? F_RP / 4 : (kind == 4 ? F_RF / 4 + src * 128 : F_PG / 4 + src * 64)));
        };
        auto absorb = [&](unsigned kind, unsigned f0, unsigned f1) {
            if (kind == 5) { const bool use = lane < WPL * WPW; while (pgmin < 7 && __all(!use || f0 >= call * 8u + pgmin + 1)) ++pgmin; pgmin = rfl(pgmin); }
            else { m0 = __ballot(f0 >= call); m1 = __ballot(f1 >= call); mkind = kind; }
        };
        unsigned t = skip + wg * WPW + wid;
        for (;;) {
            ok = rfl(ok) != 0u; mkind = rfl(mkind); pgmin = rfl(pgmin); posted = rfl(posted); pend = rfl(pend);
            m0 = rfl((unsigned)m0) | ((unsigned long long)rfl((unsigned)(m0 >> 32)) << 32); m1 = rfl((unsigned)m1) | ((unsigned long long)rfl((unsigned)(m1 >> 32)) << 32);
            if (flt && t >= NFLOAT) break;
            const unsigned kind = kind_of(t, flt), idx = idx_of(t, flt, kind);
            const unsigned tt = flt ? t : t - (kind == 1 ? 0u : n0);
            if (e != 1) { if (flt) topost = 7; else if (kind == 4) topost = group_of(idx); }
            if (!is_ready(kind, idx)) {
                WAIT_V(0); publish();
                const unsigned* fp = flags_of(kind);
                const unsigned long long t0 = __builtin_amdgcn_s_memrealtime();
                for (;;) {
                    const unsigned f0 = ld_flag(fp + lane), f1 = ld_flag(fp + 64 + lane);
                    const unsigned tm = __hip_atomic_load(tmo, __ATOMIC_RELAXED, __HIP_MEMORY_SCOPE_AGENT);
                    absorb(kind, f0, f1);
                    if (is_ready(kind, idx)) break;
                    if (rfl(tm) == call || __builtin_amdgcn_s_memrealtime() - t0 > DEADLINE) {
                        __hip_atomic_store(tmo, call, __ATOMIC_RELAXED, __HIP_MEMORY_SCOPE_AGENT);
                        ok = false; break;
                    }
                    __builtin_amdgcn_s_sleep(4);
                }
            }
            ok = rfl(ok) != 0u; mkind = rfl(mkind); pgmin = rfl(pgmin);
            unsigned so, dof, sstr = 1024, dstr = 1024, fl = 0;
            if (kind == 1) {
                const unsigned step = t >> 6, isw = (t >> 5) & 1, u = t & 31;
                const unsigned blk = isw ? col_of(step, hh) : step;
                so = (isw ? OFF_EW + (qj * 16 + blk) * EBLK_BYTES : OFF_EX + (qi * 16 + blk) * EBLK_BYTES) + (hh * 32 + u) * UNIT_BYTES + lane * 16;
                dof = (isw ? OFF_WS : OFF_XS) + blk * BLK_BYTES + (16 + (e - 2) * 8) * KT_BYTES + u * UNIT_BYTES + lane * 16;
                fl = (F_BLK + (isw * 16 + blk) * 1024) / 4 + u * 8 + (e - 2);
            } else if (kind == 3) {
                so = OFF_PL + tt * UNIT_BYTES + lane * 16; dof = so - OFF_PL + OFF_PB;
                fl = (F_FP + idx * 128) / 4 + (tt % TU);
            } else {
                const unsigned rj = ((unsigned)__builtin_amdgcn_readlane((int)trj, (int)(idx & 63)) >> ((idx >> 6) * 8)) & 255u;
                dof = own_off(flt ? rank ^ 1u : src) + rj_off(rj) + ((tt % TU) * 16 + (lane >> 5)) * 16384u + (lane & 31) * 16;
                dstr = 32768;
                if (kind == 4) so = OFF_FT + tt * UNIT_BYTES + lane * 16; else { so = dof; sstr = 32768; }
            }
            const rsrc_t rs = kind == 5 ? mkrs(sout, ok ? OUT_BYTES : 0u) : mkrs(sws, ok ? WS_USED : 0u);
            const rsrc_t rd = kind >= 4 ? mkrs(mout, OUT_BYTES) : mkrs(myws, WS_USED);
            const u32x4 nanv = (u32x4){0x7fc07fc0u, 0x7fc07fc0u, 0x7fc07fc0u, 0x7fc07fc0u};
            u32x4 va = ld16c(rs, so), vb;
            asm volatile("" ::: "memory");
            unsigned nt = 0;
            if (lane == 0) nt = draw(flt ? fqueue : queue);
            WAIT_B(0); publish();
            unsigned tn = rfl(nt) - (flt ? fbase : lbase);
            bool fn = flt;
            if (flt && tn >= NFLOAT) {
                u32x4 vt[7];
#pragma unroll
                for (int i = 0; i < 7; ++i) vt[i] = ld16c(rs, so + (i + 1) * sstr);
                st16(ok ? va : nanv, rd, dof);
#pragma unroll
                for (int i = 0; i < 7; ++i) st16(ok ? vt[i] : nanv, rd, dof + (i + 1) * dstr);
                break;
            }
            if (!flt && tn >= total) { fn = true; if (lane == 0) nt = draw(fqueue); }
            unsigned kn = 0, f0 = 0, f1 = 0;
#pragma unroll
            for (int i = 0; i < 8; i += 2) {
                vb = ld16c(rs, so + (i + 1) * sstr);
                if (i == 6) {
                    if (fn && !flt) tn = rfl(nt) - fbase;
                    tn = rfl(tn);
                    if (!fn || tn < NFLOAT) {
                        const unsigned k2 = kind_of(tn, fn);
                        if (!is_ready(k2, idx_of(tn, fn, k2))) { kn = k2; const unsigned* fp = flags_of(k2); f0 = ld_flag(fp + lane); f1 = ld_flag(fp + 64 + lane); }
                    }
                }
                asm volatile("" ::: "memory");
                st16(ok ? va : nanv, rd, dof + i * dstr);
                WAIT_B(1);
                if (i + 2 < 8) va = ld16c(rs, so + (i + 2) * sstr);
                asm volatile("" ::: "memory");
                st16(ok ? vb : nanv, rd, dof + (i + 1) * dstr);
                if (i + 2 < 8) WAIT_B(1);
            }
            if (kn) absorb(kn, f0, f1);
            pend = fl;
            t = rfl(tn); flt = rfl(fn) != 0u;
        }
        if (e != 1) topost = 7;
        if (pend || topost > posted) { WAIT_V(0); publish(); }
        STAMP(4);
        return;
    }

    const unsigned g = bid - NCOMM;
    const int wr = wid >> 2, wc = wid & 3, fr = lane & 15, fq = lane >> 4;
    const unsigned voff = tid * 16;
    const unsigned ldsw = (unsigned)wid * 1024u;
    int aoff[2], boff[2];
#pragma unroll
    for (int q = 0; q < 2; ++q) {
        aoff[q] = wr * 8192 + (fr & 7) * 1024 + (fr >> 3) * 128 + (((fq ^ (fr & 7)) ^ (q << 2)) * 16);
        boff[q] = (wc >> 1) * 8192 + (fr & 3) * 1024 + (4 * (wc & 1) + (fr >> 2)) * 128 + ((fq ^ (fr & 3) ^ ((((fr >> 3) & 1) ^ q) << 2)) * 16);
    }
#define SA(b, h) (((b) * 2 + (h)) * HT_BYTES)
#define SB(b, h) ((4 + (b) * 2 + (h)) * HT_BYTES)
#define STAGE(bufoff, gbase) do { _Pragma("unroll") for (int _i = 0; _i < 2; ++_i) \
        __builtin_amdgcn_global_load_lds((const unsigned*)((const char*)(gbase) + voff + _i * 8192), (LAS unsigned*)(lds + (bufoff) + ldsw + _i * 8192), 16, 0, 0); } while (0)
#define LDA(dst, b, h) do { _Pragma("unroll") for (int m = 0; m < 4; ++m) _Pragma("unroll") for (int k = 0; k < 2; ++k) dst[m][k] = *(const LAS bf16x8*)(lds + SA(b, h) + aoff[(k ^ m) & 1] + m * 256); } while (0)
#define LDB(dst, b, h) do { _Pragma("unroll") for (int n = 0; n < 2; ++n) _Pragma("unroll") for (int k = 0; k < 2; ++k) dst[n][k] = *(const LAS bf16x8*)(lds + SB(b, h) + boff[(k ^ n) & 1] + n * 4096); } while (0)
#define MMA(ai, bj, At, Bt) do { __builtin_amdgcn_s_setprio(1); _Pragma("unroll") for (int m = 0; m < 4; ++m) _Pragma("unroll") for (int n = 0; n < 2; ++n) _Pragma("unroll") for (int k = 0; k < 2; ++k) \
        acc[ai][bj][m][n] = __builtin_amdgcn_mfma_f32_16x16x32_bf16(Bt[n][k], At[m][k], acc[ai][bj][m][n], 0, 0, 0); __builtin_amdgcn_s_setprio(0); } while (0)

    for (int round = 0; round < 4; ++round) {
        const unsigned task = kTask[g * 4 + round];
        if (task == 0xffffffffu) break;
        const unsigned ti = task & 127, own = (task >> 7) & 1, kp = (task >> 18) & 7, lgw = (task >> 21) & 3, sl = (task >> 23) & 63;
        const int k0 = ((task >> 8) & 31) * 4, k1 = ((task >> 13) & 31) * 4;
        const unsigned rj = kTileRJ[ti], r = rj >> 3, c = ((own ^ hh ^ 1) & 1) * 8 + (rj & 7);
        auto wait_remote = [&]() {
            const unsigned* fp = myfl + (F_BLK + (lane < 32 ? r : 16 + c) * 1024) / 4 + (lane & 31);
            const bool use = (lane & 7) < 6;
            (void)wave_wait<1>(fp + 7 * 32, use, call, call, tmo);
            (void)wave_wait<7>(fp, use, call, call, tmo);
            STAMPM(0);
        };
        if (wid == 0) {
            if (k0 < 16) (void)wave_wait<1>(myfl + (F_OWN + (lane < 8 ? r : 16 + c) * 128) / 4 + (lane & 7), lane < 16, call, call, tmo);
            else wait_remote();
            __builtin_amdgcn_fence(__ATOMIC_ACQUIRE, "");
            WAIT_V(0);
        }
        wg_barrier();
        f32x4 acc[2][2][4][2];
#pragma unroll
        for (int a = 0; a < 2; ++a)
#pragma unroll
            for (int b = 0; b < 2; ++b)
#pragma unroll
                for (int m = 0; m < 4; ++m)
#pragma unroll
                    for (int n = 0; n < 2; ++n) acc[a][b][m][n] = (f32x4){0.f, 0.f, 0.f, 0.f};
        bf16x8 At[4][2], B0[2][2], B1[2][2];
        const char* eA = myws + OFF_EX + (qi * 16 + r) * EBLK_BYTES;
        const char* eB = myws + OFF_EW + (qj * 16 + c) * EBLK_BYTES;
        const char* cA = myws + OFF_XS + r * BLK_BYTES;
        const char* cB = myws + OFF_WS + c * BLK_BYTES;
        {
            const char* a0 = (k0 < 16 ? eA : cA) + (size_t)k0 * KT_BYTES; const char* b0 = (k0 < 16 ? eB : cB) + (size_t)k0 * KT_BYTES;
            STAGE(SB(0, 0), b0); STAGE(SA(0, 0), a0); STAGE(SB(0, 1), b0 + HT_BYTES); STAGE(SA(0, 1), a0 + HT_BYTES);
            if (wr == 1) BAR;
            WAIT_V(4); BAR;
            STAGE(SB(1, 0), b0 + KT_BYTES); STAGE(SA(1, 0), a0 + KT_BYTES); STAGE(SB(1, 1), b0 + HT_BYTES + KT_BYTES);
            WAIT_V(6); BAR;
        }
        for (int t = k0; t < k1; t += 2) {
            const int t2 = t + 2 < k1 ? t + 2 : k0;
            const char* a1 = (t < 16 ? eA : cA) + (size_t)(t + 1) * KT_BYTES;
            const char* a2 = (t2 < 16 ? eA : cA) + (size_t)t2 * KT_BYTES; const char* b2 = (t2 < 16 ? eB : cB) + (size_t)t2 * KT_BYTES;
            const char* a3 = a2 + KT_BYTES; const char* b3 = b2 + KT_BYTES;
            if (t == 14 && k1 > 16) {
                if (wid == 0) {
                    wait_remote();
                    __builtin_amdgcn_fence(__ATOMIC_ACQUIRE, "");
                    WAIT_V(0);
                }
                asm volatile("" ::: "memory"); BAR; asm volatile("" ::: "memory");
            }
            LDB(B0, 0, 0); SCHED; LDA(At, 0, 0); STAGE(SA(1, 1), a1 + HT_BYTES);
            WAIT_L(8); BAR; WAIT_L(0); MMA(0, 0, At, B0); BAR; SCHED;
            LDB(B1, 0, 1); STAGE(SB(0, 0), b2);
            BAR; WAIT_L(0); MMA(0, 1, At, B1); BAR;
            LDA(At, 0, 1); STAGE(SA(0, 0), a2);
            BAR; WAIT_L(0); MMA(1, 0, At, B0); BAR; SCHED;
            STAGE(SB(0, 1), b2 + HT_BYTES);
            WAIT_V(6); BAR; MMA(1, 1, At, B1); BAR;
            LDB(B0, 1, 0); SCHED; LDA(At, 1, 0); STAGE(SA(0, 1), a2 + HT_BYTES);
            WAIT_L(8); BAR; WAIT_L(0); MMA(0, 0, At, B0); BAR; SCHED;
            LDB(B1, 1, 1); STAGE(SB(1, 0), b3);
            BAR; WAIT_L(0); MMA(0, 1, At, B1); BAR;
            LDA(At, 1, 1); STAGE(SA(1, 0), a3);
            BAR; WAIT_L(0); MMA(1, 0, At, B0); BAR; SCHED;
            STAGE(SB(1, 1), b3 + HT_BYTES);
            WAIT_V(6); BAR; MMA(1, 1, At, B1); BAR;
        }
        WAIT_V(0);
        if (wr == 0) BAR;
        BAR;
        STAMPM(1);
        if (!lgw && !own) {
            const bool bad = rfl(__hip_atomic_load(tmo, __ATOMIC_RELAXED, __HIP_MEMORY_SCOPE_AGENT)) == call;
            const unsigned base = OFF_PL + ti * TILE_BYTES;
#pragma unroll
            for (int ai = 0; ai < 2; ++ai)
#pragma unroll
                for (int m = 0; m < 4; ++m)
#pragma unroll
                    for (int bj = 0; bj < 2; ++bj) {
                        const f32x4 v0 = acc[ai][bj][m][0], v1 = acc[ai][bj][m][1];
                        u32x4 o; o.x = cvt_pk_bf16(v0[0], v0[1]); o.y = cvt_pk_bf16(v0[2], v0[3]); o.z = cvt_pk_bf16(v1[0], v1[1]); o.w = cvt_pk_bf16(v1[2], v1[3]);
                        if (bad) o = (u32x4){0x7fc07fc0u, 0x7fc07fc0u, 0x7fc07fc0u, 0x7fc07fc0u};
                        st16(o, rmy, voff, base + ((ai * 4 + m) * 2 + bj) * 8192);
                    }
            WAIT_V(0); wg_barrier();
            if (tid == 0) st_flag((unsigned*)(P.ws[rank ^ 1u] + OFF_FL + F_RP) + ti, call);
        }
        if (own) {
            if (wid == 0) {
                (void)wave_wait<1>(myfl + (F_FP + ti * 128) / 4 + (lane % TU), true, call, call, tmo);
                __builtin_amdgcn_fence(__ATOMIC_ACQUIRE, "");
                WAIT_V(0);
            }
            wg_barrier();
            STAMPM(2);
            const bool bad = rfl(__hip_atomic_load(tmo, __ATOMIC_RELAXED, __HIP_MEMORY_SCOPE_AGENT)) == call;
            const rsrc_t rmo = mkrs(P.out[rank], OUT_BYTES);
            const unsigned pbase = OFF_PB + ti * TILE_BYTES;
            int lz = lane; asm volatile("" : "+v"(lz));
            const unsigned obase = own_off(rank) + rj_off(rj) + wr * 64 * 16384u + wc * 64, ov = (lz & 15) * 16384u + (lz >> 4) * 16;
            const unsigned fn = rfl(kP2F[ti]);
            const unsigned fbase = OFF_FT + fn * TILE_BYTES + wr * 64 * 512u + wc * 64, fv = (lz & 15) * 512u + (lz >> 4) * 16;
            const rsrc_t rpb = mkrs(myws, bad ? 0u : WS_USED);
            u32x4 pv[16];
#pragma unroll
            for (int q = 0; q < 16; ++q) pv[q] = ld16c(rpb, voff, pbase + q * 8192);
#pragma unroll
            for (int q = 0; q < 16; ++q) {
                const int ai = q >> 3, m = (q >> 1) & 3, bj = q & 1;
                const f32x4 v0 = acc[ai][bj][m][0], v1 = acc[ai][bj][m][1]; const u32x4 p = pv[q];
                u32x4 o;
                o.x = cvt_pk_bf16(silu(v0[0] + __uint_as_float(p.x << 16)), silu(v0[1] + __uint_as_float(p.x & 0xffff0000u)));
                o.y = cvt_pk_bf16(silu(v0[2] + __uint_as_float(p.y << 16)), silu(v0[3] + __uint_as_float(p.y & 0xffff0000u)));
                o.z = cvt_pk_bf16(silu(v1[0] + __uint_as_float(p.z << 16)), silu(v1[1] + __uint_as_float(p.z & 0xffff0000u)));
                o.w = cvt_pk_bf16(silu(v1[2] + __uint_as_float(p.w << 16)), silu(v1[3] + __uint_as_float(p.w & 0xffff0000u)));
                if (bad) o = (u32x4){0x7fc07fc0u, 0x7fc07fc0u, 0x7fc07fc0u, 0x7fc07fc0u};
                st16(o, rmy, fv, fbase + (ai * 128 + m * 16) * 512u + bj * 256);
                pv[q] = o;
            }
            WAIT_V(0); wg_barrier();
            if (tid < NR && (unsigned)tid != rank) st_flag((unsigned*)(P.ws[tid] + OFF_FL + F_RF) + rank * 128 + fn, call);
#pragma unroll
            for (int q = 0; q < 16; ++q) st16(pv[q], rmo, ov, obase + ((q >> 3) * 128 + ((q >> 1) & 3) * 16) * 16384u + (q & 1) * 256);
            STAMPM(3);
        }
        if (lgw) {
            const unsigned W = 1u << lgw;
            const unsigned sbase = OFF_SL + sl * 8 * (2 * TILE_BYTES);
#pragma unroll
            for (int q = 0; q < 32; ++q) st16(__builtin_bit_cast(u32x4, acc[q >> 4][(q >> 1) & 1][(q >> 2) & 3][q & 1]), rmy, voff, sbase + kp * (2 * TILE_BYTES) + q * 8192);
            WAIT_V(0); wg_barrier();
            unsigned* fs = myfl + (F_SL + sl * 128) / 4;
            if (tid == 0) st_flag(fs + kp, call);
            if (wid == 0) (void)wave_wait<1>(fs + lane, (unsigned)lane < W, call, call, tmo);
            wg_barrier();
            const bool bad = rfl(__hip_atomic_load(tmo, __ATOMIC_RELAXED, __HIP_MEMORY_SCOPE_AGENT)) == call;
            const unsigned q0 = kp * (32u >> lgw);
#define ACC(q) acc[(q) >> 4][((q) >> 1) & 1][((q) >> 2) & 3][(q) & 1]
#pragma unroll
            for (int z = 0; z < 32; ++z) ACC(z) = __builtin_bit_cast(f32x4, ld16c(rmy, voff, sbase + (z & (W - 1)) * (2 * TILE_BYTES) + (q0 + (z >> lgw)) * 8192));
            f32x4 s4[8];
#pragma unroll
            for (int z = 0; z < 8; ++z) s4[z] = (ACC(4 * z) + ACC(4 * z + 1)) + (ACC(4 * z + 2) + ACC(4 * z + 3));
#undef ACC
            if (lgw == 3) {
#pragma unroll
                for (int z = 0; z < 4; ++z) s4[z] = s4[2 * z] + s4[2 * z + 1];
            }
            const unsigned base = OFF_PL + ti * TILE_BYTES + (q0 >> 1) * 8192;
#pragma unroll
            for (int z = 0; z < 4; ++z) {
                if (z < 2 || lgw == 2) {
                    const f32x4 v0 = s4[2 * z], v1 = s4[2 * z + 1];
                    u32x4 o; o.x = cvt_pk_bf16(v0[0], v0[1]); o.y = cvt_pk_bf16(v0[2], v0[3]); o.z = cvt_pk_bf16(v1[0], v1[1]); o.w = cvt_pk_bf16(v1[2], v1[3]);
                    if (bad) o = (u32x4){0x7fc07fc0u, 0x7fc07fc0u, 0x7fc07fc0u, 0x7fc07fc0u};
                    st16(o, rmy, voff, base + z * 8192);
                }
            }
            WAIT_V(0); wg_barrier();
            if (tid == 0 && draw(fs + 16) + 1u == call * W) st_flag((unsigned*)(P.ws[rank ^ 1u] + OFF_FL + F_RP) + ti, call);
            STAMPT(ti);
        }
    }
}

}

extern "C" void kernel_launch(int rank, int nranks, ncclComm_t comm,
                              hipStream_t stream,
                              void* const* d_in, const int* in_sizes, int n_in,
                              void* d_out, int out_size,
                              const int* in_dtypes, int out_dtype,
                              void* const* d_out_all,
                              void* const* d_ws_all, size_t ws_bytes) {
    if (nranks != NR || (unsigned)rank >= (unsigned)NR || ws_bytes < (size_t)WS_USED || n_in < 2) {
        fprintf(stderr, "unsupported configuration\n");
        return;
    }
    Params P;
    P.x = (const char*)d_in[0]; P.w = (const char*)d_in[1]; P.rank = rank;
    for (int p = 0; p < NR; ++p) { P.out[p] = (char*)d_out_all[p]; P.ws[p] = (char*)d_ws_all[p]; }
    HIP_CHECK(hipFuncSetAttribute((const void*)gemm_ar_kernel, hipFuncAttributeMaxDynamicSharedMemorySize, LDS_BYTES));
    hipLaunchKernelGGL(gemm_ar_kernel, dim3(NWG), dim3(512), LDS_BYTES, stream, P);
    HIP_CHECK(hipGetLastError());
}
```
